```python
import numpy as np

kernel_name = "gpu_gemm_rs_m4096_k4096_n16384_bf16_relu"


ABSMAX_RTOL = 0.008

COMPUTE_DTYPE = "bfloat16"
EPILOGUE = "relu"
IN_QUANT = None
OUT_QUANT = None
IN_DTYPES = {"x": "bfloat16", "w_mat": "bfloat16"}
WORLD_SIZE = 8

def _to_bf16(x):
    u = np.ascontiguousarray(x, dtype=np.float32).view(np.uint32)
    return ((u + 0x7FFF + ((u >> 16) & 1)) & 0xFFFF0000).view(np.float32)

def setup_inputs():

    x = _to_bf16(np.random.randn(4096, 4096).astype(np.float32))
    w_mat = _to_bf16((np.random.randn(4096, 16384) / np.sqrt(4096)).astype(np.float32))
    return {"x": x, "w_mat": w_mat}

def _epilogue(y):
    return np.maximum(y, 0.0).astype(np.float32)

def reference(x, w_mat):
    y = (x @ w_mat).astype(np.float32)
    return _epilogue(y)
```

```cpp
#include <hip/hip_runtime.h>
#include <rccl/rccl.h>
#include <cstdio>
#include <cstdint>

#define HIP_CHECK(expr) do { hipError_t _e = (expr); if (_e != hipSuccess) { \
    fprintf(stderr, "HIP error %s at %s:%d\n", hipGetErrorString(_e), __FILE__, __LINE__); \
    return; } } while (0)

namespace {

#define LAS __attribute__((address_space(3)))
#define GAS __attribute__((address_space(1)))
typedef GAS char gchar;
typedef GAS unsigned gu32;
typedef GAS const unsigned short gbf16;
typedef short bf16x8 __attribute__((ext_vector_type(8)));
typedef float f32x4 __attribute__((ext_vector_type(4)));
typedef unsigned u32x4 __attribute__((ext_vector_type(4)));

constexpr int NR = 8;
constexpr int KL = 512;
constexpr int NG = 16384;
constexpr int NL = NG / NR;
constexpr int ML = 512;
constexpr int NRB = 16, NCB = 8;
constexpr int NKT = 64;
constexpr unsigned PIECE = 32768;
constexpr int ITEMS_PER_ROUND = NRB + NCB, ITEMS_PER_LINK = 8 * ITEMS_PER_ROUND;

constexpr size_t OFF_X = 0;
constexpr size_t OFF_W = OFF_X + (size_t)NKT * NRB * PIECE;
constexpr size_t OFF_FLAG = OFF_W + (size_t)NKT * NCB * PIECE;
constexpr size_t OFF_DONE = OFF_FLAG + (size_t)NKT * ITEMS_PER_ROUND * 128;
constexpr size_t OFF_TMO = OFF_DONE + NR * 128;
constexpr size_t OFF_QUEUE = OFF_TMO + 128;
constexpr size_t OFF_TICKET = OFF_QUEUE + NR * 128;
constexpr size_t OFF_DBG = OFF_TICKET + 256 * 128;
constexpr size_t WS_NEED = OFF_DBG + 512 * 1024;
constexpr unsigned OUT_BYTES = (unsigned)ML * NG * 2;

#define NSEND 77
#define SW 1
constexpr int NGEMM = NRB * NCB;
constexpr int NHELP = 48;
constexpr int GRID = NSEND + NGEMM + NHELP;
constexpr unsigned UNITS = 4 * ITEMS_PER_LINK;
constexpr unsigned WPL = NSEND * SW / 7;
constexpr unsigned PER_CALL = UNITS + WPL;
static_assert(NSEND * SW % 7 == 0 && GRID <= 256 && 2 * WPL <= UNITS, "grid");

constexpr unsigned HTB = 16384;
constexpr unsigned LDS_STAGE = 8 * HTB;
constexpr unsigned LDS_BYTES = LDS_STAGE + 1024;
constexpr unsigned long long DEADLINE = 30000000ull;

struct Args {
    gbf16* x;
    gbf16* w;
    gchar* ws[NR];
    gchar* out[NR];
    int rank;
    int pad;
};

__device__ __forceinline__ unsigned cvt_pk_bf16(float lo, float hi) {
    typedef float f32x2 __attribute__((ext_vector_type(2)));
    typedef __bf16 bf16x2 __attribute__((ext_vector_type(2)));
    return __builtin_bit_cast(unsigned, __builtin_convertvector((f32x2){lo, hi}, bf16x2));
}
__device__ __forceinline__ gchar* uniform(gchar* p) {
    const unsigned long long v = (unsigned long long)p;
    const unsigned lo = __builtin_amdgcn_readfirstlane((unsigned)v), hi = __builtin_amdgcn_readfirstlane((unsigned)(v >> 32));
    return (gchar*)(((unsigned long long)hi << 32) | lo);
}
typedef __amdgpu_buffer_rsrc_t rsrc_t;
__device__ __forceinline__ rsrc_t make_rsrc(gchar* base, unsigned bytes) { return __builtin_amdgcn_make_buffer_rsrc((char*)uniform(base), 0, bytes, 0x00020000); }
__device__ __forceinline__ void st16_sys(rsrc_t r, unsigned voff, unsigned soff, u32x4 v) { __builtin_amdgcn_raw_buffer_store_b128(v, r, voff + soff, 0, 17); }
__device__ __forceinline__ void drain() { asm volatile("s_waitcnt vmcnt(0)" ::: "memory"); __builtin_amdgcn_s_waitcnt(0x0F70); }
__device__ __forceinline__ unsigned ld_sys(gu32* p) { return __hip_atomic_load(p, __ATOMIC_RELAXED, __HIP_MEMORY_SCOPE_SYSTEM); }
__device__ __forceinline__ unsigned ld_agent(gu32* p) { return __hip_atomic_load(p, __ATOMIC_RELAXED, __HIP_MEMORY_SCOPE_AGENT); }

#define POLL_SLEEP 8
__device__ __forceinline__ void wait_flags(gu32* p, unsigned want, gu32* tmo, unsigned call) {
    const unsigned long long t0 = __builtin_amdgcn_s_memrealtime();
    unsigned polls = 0;
    while (__ballot(ld_sys(p) < want) != 0ull) {
        if ((++polls & 63u) == 0u &&
            (__builtin_amdgcn_readfirstlane(ld_agent(tmo)) == call || __builtin_amdgcn_s_memrealtime() - t0 > DEADLINE)) {
            __hip_atomic_store(tmo, call, __ATOMIC_RELAXED, __HIP_MEMORY_SCOPE_AGENT);
            break;
        }
        __builtin_amdgcn_s_sleep(POLL_SLEEP);
    }
    asm volatile("s_waitcnt vmcnt(0) lgkmcnt(0)" ::: "memory");
    __builtin_amdgcn_s_waitcnt(0x0070);
}

#define GST(voff, v, sbase, imm) asm volatile("global_store_dwordx4 %0, %1, %2 offset:" #imm " sc0 sc1\n\ts_nop 1" :: "v"(voff), "v"(v), "s"(sbase) : "memory")
#define GLDS(g, l) __builtin_amdgcn_global_load_lds((gu32*)(g), (LAS unsigned*)(l), 16, 0, 0)
__device__ __forceinline__ void load_unit(const Args& a, unsigned u, int dst, int lane, LAS unsigned char* buf) {
    const int it = u >> 2, wq = u & 3;
    const int j = it / ITEMS_PER_ROUND, r = it % ITEMS_PER_ROUND, g = lane >> 4, c = lane & 15;
    if (r < NCB) {
        gchar* p = (gchar*)(a.w + (size_t)(64 * j + 32 * (wq & 1)) * NG + NL * dst + 256 * r + 32 * (wq >> 1));
        const unsigned o = (unsigned)(8 * g * NG + 64 * (c >> 2) + 8 * (c & 3)) * 2u;
#pragma unroll
        for (int t = 0; t < 8; ++t) GLDS(p + t * (NG * 2) + o, buf + t * 1024);
    } else {
        const int rb = r - NCB;
        const unsigned o = (unsigned)(c * KL + 8 * g) * 2u;
#pragma unroll
        for (int h = 0; h < 2; ++h) {
            gchar* p = (gchar*)(a.x + (size_t)(ML * ((2 * wq + h + rb) & 7) + 32 * rb) * KL + 64 * j);
#pragma unroll
            for (int t = 0; t < 4; ++t) GLDS(p + (t >> 1) * (16 * KL * 2) + (t & 1) * 64 + o, buf + (4 * h + t) * 1024);
        }
    }
}
__device__ __forceinline__ void store_unit(const Args& a, unsigned u, gchar* wsd, int lane, LAS unsigned char* buf) {
    const int it = u >> 2, wq = u & 3;
    const int j = it / ITEMS_PER_ROUND, r = it % ITEMS_PER_ROUND, kt = a.rank * 8 + j;
    const unsigned o = lane * 16u;
    u32x4 v[8];
#pragma unroll
    for (int t = 0; t < 8; ++t) v[t] = *(LAS u32x4*)(buf + t * 1024 + o);
    if (r < NCB) {
        gchar* p = wsd + OFF_W + (size_t)(kt * NCB + r) * PIECE + wq * 8192;
        gchar* p2 = p + 4096;
        u32x4 t[8];
#pragma unroll
        for (int i = 0; i < 8; ++i) {
            const unsigned sel = (i & 1) ? 0x07060302u : 0x05040100u;
#pragma unroll
            for (int e = 0; e < 4; ++e) t[i][e] = __builtin_amdgcn_perm(v[2 * e + 1][i >> 1], v[2 * e][i >> 1], sel);
        }
        GST(o, t[0], p, 0); GST(o ^ 64u, t[1], p, 1024); GST(o ^ 128u, t[2], p, 2048); GST(o ^ 192u, t[3], p, 3072);
        GST(o, t[4], p2, 0); GST(o ^ 64u, t[5], p2, 1024); GST(o ^ 128u, t[6], p2, 2048); GST(o ^ 192u, t[7], p2, 3072);
    } else {
        gchar* p = wsd + OFF_X + (size_t)(kt * NRB + (r - NCB)) * PIECE + wq * 8192;
        gchar* p2 = p + 4096;
        GST(o, v[0], p, 0); GST(o, v[1], p, 1024); GST(o, v[2], p, 2048); GST(o, v[3], p, 3072);
        GST(o, v[4], p2, 0); GST(o, v[5], p2, 1024); GST(o, v[6], p2, 2048); GST(o, v[7], p2, 3072);
    }
}
#define WAIT_VM(n) asm volatile("s_waitcnt vmcnt(" #n ")" ::: "memory")
__device__ __forceinline__ void draw(unsigned& f, gchar* head) {
    unsigned long long keep;
    asm volatile("s_mov_b64 %1, exec\n\ts_mov_b64 exec, 1\n\tglobal_atomic_add %0, %2, %3, %4 sc0\n\ts_mov_b64 exec, %1"
                 : "=&v"(f), "=&s"(keep) : "v"(0u), "v"(1u), "s"(head) : "memory");
}
__device__ __forceinline__ gu32* flag_of(gchar* wsd, int src, unsigned u) { return (gu32*)(wsd + OFF_FLAG + (size_t)(src * ITEMS_PER_LINK + (u >> 2)) * 128 + (u & 3) * 4); }

__device__ __forceinline__ int kt_of(int s, int rank) {
    if (s < 8) return rank * 8 + s;
    const int u = s - 8, j = u / 7, l = u - 7 * j;
    return ((rank + 1 + l) & 7) * 8 + j;
}

#define STAMP_AT(i) do {} while (0)

__global__ __launch_bounds__(512, 2) void gemm_rs_kernel(const Args a) {
    extern __shared__ __attribute__((aligned(16))) unsigned char smem[];
    LAS unsigned char* lds = (LAS unsigned char*)smem;
    LAS unsigned* ctl = (LAS unsigned*)(lds + LDS_STAGE);
    const int tid = threadIdx.x, wid = __builtin_amdgcn_readfirstlane(tid >> 6), lane = tid & 63;
    const int rank = a.rank;
    int bid = blockIdx.x;
    asm volatile("" : "+s"(bid));
    gchar* ws = a.ws[rank];
    gu32* tmo = (gu32*)(ws + OFF_TMO);
    if (bid < NSEND) {
        if (wid >= SW) return;
        const unsigned gw = bid * SW + wid, link = gw % 7;
        const int dst = (rank + 1 + link) & 7;
        gchar* wsd = a.ws[dst];
        gchar* head = ws + OFF_QUEUE + link * 128;
        unsigned call = 0, f, prev = UNITS, cur = gw / 7, nxt = cur + WPL, bsel = 0;
        LAS unsigned char* buf = lds + wid * 16384;
        auto publish = [&](unsigned u) {
            if (lane == 0) {
                __hip_atomic_store(flag_of(wsd, rank, u), call, __ATOMIC_RELAXED, __HIP_MEMORY_SCOPE_SYSTEM);
            }
        };
        load_unit(a, cur, dst, lane, buf);
        load_unit(a, nxt, dst, lane, buf + 8192);
        draw(f, head);
        WAIT_VM(9);
        do {
            store_unit(a, cur, wsd, lane, buf + bsel);
            asm volatile("s_waitcnt vmcnt(8)" : "+v"(f) :: "memory");
            const unsigned tk = __builtin_amdgcn_readfirstlane(f), nn = 2 * WPL + tk % PER_CALL;
            call = tk / PER_CALL + 1u;
            if (prev < UNITS) publish(prev);
            if (nn < UNITS) load_unit(a, nn, dst, lane, buf + bsel);
            draw(f, head);
            prev = cur; cur = nxt; nxt = nn; bsel ^= 8192u;
        } while (cur < UNITS);
        asm volatile("s_waitcnt vmcnt(0)" : "+v"(f) :: "memory");
        publish(prev);
        if (gw == 0) {
            const int src = (rank + 1 + (lane % 7)) & 7;
            wait_flags((gu32*)(ws + OFF_DONE + src * 128), call * NGEMM, tmo, call);
            if (lane == 0 && ld_agent(tmo) == call) *(GAS unsigned short*)(a.out[rank] + (size_t)rank * NL * 2) = 0x7fc0;
            STAMP_AT(1);
        }
        return;
    }

    if (tid == 0)
        ctl[0] = __hip_atomic_fetch_add((gu32*)(ws + OFF_TICKET + (size_t)bid * 128), 1u, __ATOMIC_RELAXED, __HIP_MEMORY_SCOPE_AGENT) + 1u;
    if (bid >= NSEND + NGEMM) {
        const unsigned u0 = (bid - NSEND - NGEMM) * 8 + wid, u1 = u0 + NHELP * 8;
        static_assert(2 * NHELP * 8 == UNITS, "two units per helper wave");
        LAS unsigned char* buf = lds + wid * 16384;
        load_unit(a, u0, rank, lane, buf);
        load_unit(a, u1, rank, lane, buf + 8192);
        WAIT_VM(8);
        store_unit(a, u0, ws, lane, buf);
        WAIT_VM(8);
        store_unit(a, u1, ws, lane, buf + 8192);
        drain();
        __syncthreads();
        if (lane < 2)
            __hip_atomic_store(flag_of(ws, rank, lane ? u1 : u0), ctl[0], __ATOMIC_RELAXED, __HIP_MEMORY_SCOPE_SYSTEM);
        STAMP_AT(1);
        return;
    }

    const int gb = bid - NSEND;
    const int xcd = bid & 7, idx = gb >> 3;
    const int rb = (xcd >> 1) * 4 + (idx & 3), cb = (xcd & 1) * 4 + (idx >> 2);
    __syncthreads();
    const unsigned call = __builtin_amdgcn_readfirstlane(ctl[0]);
    STAMP_AT(1);

    const int wr = wid >> 2, wc = wid & 3, fr = lane & 15, fq = lane >> 4;
    const unsigned voff = (unsigned)tid * 16u;
    const unsigned ldsw = (unsigned)wid * 1024u;
    const int aoff = wr * 8192 + lane * 16;
    const int boff = (fr & 3) * 1024 + fq * 256 + ((((wc ^ (fr & 3)) << 2) | (fr >> 2)) << 4);
    gchar* xbase = ws + OFF_X + (size_t)rb * PIECE;
    gchar* wbase = ws + OFF_W + (size_t)cb * PIECE;
    const unsigned fl_r = ((lane & 4) ? (unsigned)cb : (unsigned)(NCB + rb)) * 128u + (lane & 3) * 4u;
#define XP(kt) (xbase + (size_t)(kt) * (NRB * PIECE))
#define WP(kt) (wbase + (size_t)(kt) * (NCB * PIECE))
#define FLAGP(ka, kb) ((gu32*)(ws + OFF_FLAG + (size_t)(((lane & 8) ? (kb) : (ka)) * (ITEMS_PER_ROUND * 128u) + fl_r)))
#define SA(b, h) (((b) * 2 + (h)) * HTB)
#define SB(b, h) ((4 + (b) * 2 + (h)) * HTB)
#define STAGE(bufoff, gbase) do { _Pragma("unroll") for (int _i = 0; _i < 2; ++_i) \
        __builtin_amdgcn_global_load_lds((gu32*)((gbase) + _i * 8192 + voff), (LAS unsigned*)(lds + (bufoff) + ldsw + _i * 8192), 16, 0, LOAD_AUX); } while (0)
#define LDA(dst, b, h) do { _Pragma("unroll") for (int m = 0; m < 4; ++m) _Pragma("unroll") for (int k = 0; k < 2; ++k) \
        dst[m][k] = *(const LAS bf16x8*)(lds + SA(b, h) + aoff + m * 2048 + k * 1024); } while (0)
#define LDB(dst, b, h) do { _Pragma("unroll") for (int n = 0; n < 2; ++n) _Pragma("unroll") for (int k = 0; k < 2; ++k) \
        dst[n][k] = *(const LAS bf16x8*)(lds + SB(b, h) + boff + n * 4096 + k * 8192); } while (0)
#define MMA(ai, bj, At, Bt) do { __builtin_amdgcn_s_setprio(1); \
        _Pragma("unroll") for (int m = 0; m < 4; ++m) _Pragma("unroll") for (int n = 0; n < 2; ++n) _Pragma("unroll") for (int k = 0; k < 2; ++k) \
            acc[ai][bj][m][n] = __builtin_amdgcn_mfma_f32_16x16x32_bf16(Bt[n][k], At[m][k], acc[ai][bj][m][n], 0, 0, 0); \
        __builtin_amdgcn_s_setprio(0); } while (0)
#define WAIT_V(n) asm volatile("s_waitcnt vmcnt(" #n ")" ::: "memory")
#define WAIT_L(n) asm volatile("s_waitcnt lgkmcnt(" #n ")" ::: "memory")
#define BAR __builtin_amdgcn_s_barrier()
#define SCHED __builtin_amdgcn_sched_barrier(0)
#define LOAD_AUX 17

    f32x4 acc[2][2][4][2];
#pragma unroll
    for (int i = 0; i < 2; ++i)
#pragma unroll
        for (int j = 0; j < 2; ++j)
#pragma unroll
            for (int m = 0; m < 4; ++m)
#pragma unroll
                for (int n = 0; n < 2; ++n) acc[i][j][m][n] = (f32x4){0.f, 0.f, 0.f, 0.f};
    bf16x8 At[4][2], B0[2][2], B1[2][2];

    const int ktab = kt_of(lane, rank);
#define KT(s) __builtin_amdgcn_readlane(ktab, (s) & (NKT - 1))
    int k1 = KT(1);
    {
        const int k0 = KT(0);
        if (wid == 0) wait_flags(FLAGP(k0, k1), call, tmo, call);
        BAR;
        STAGE(SB(0, 0), WP(k0)); STAGE(SA(0, 0), XP(k0)); STAGE(SB(0, 1), WP(k0) + HTB); STAGE(SA(0, 1), XP(k0) + HTB);
        if (wr == 1) BAR;
        WAIT_V(4); BAR;
        STAGE(SB(1, 0), WP(k1)); STAGE(SA(1, 0), XP(k1)); STAGE(SB(1, 1), WP(k1) + HTB);
        WAIT_V(6); BAR;
    }
    STAMP_AT(2);
    unsigned fl = 0;
    for (int t = 0; t < NKT; t += 2) {
        const int k2 = KT(t + 2), k3 = KT(t + 3);
        if (wid == 0 && __ballot(fl < call) != 0ull) {
            wait_flags(FLAGP(k2, k3), call, tmo, call);
        }
        gchar* a1 = XP(k1); gchar* a2 = XP(k2); gchar* b2 = WP(k2); gchar* a3 = XP(k3); gchar* b3 = WP(k3);
        const int k4 = KT(t + 4), k5 = KT(t + 5);
        gu32* fp = FLAGP(k4, k5);
        LDB(B0, 0, 0); SCHED; LDA(At, 0, 0); STAGE(SA(1, 1), a1 + HTB);
        WAIT_L(8); BAR; WAIT_L(0); MMA(0, 0, At, B0); BAR; SCHED;
        LDB(B1, 0, 1); STAGE(SB(0, 0), b2);
        BAR; WAIT_L(0); MMA(0, 1, At, B1); BAR;
        LDA(At, 0, 1); STAGE(SA(0, 0), a2);
        BAR; WAIT_L(0); MMA(1, 0, At, B0); BAR; SCHED;
        STAGE(SB(0, 1), b2 + HTB);
        WAIT_V(6); BAR; MMA(1, 1, At, B1); BAR;
        if (wid == 0) asm volatile("global_load_dword %0, %1, off sc0 sc1" : "=v"(fl) : "v"(fp) : "memory");
        LDB(B0, 1, 0); SCHED; LDA(At, 1, 0); STAGE(SA(0, 1), a2 + HTB);
        WAIT_L(8); BAR; WAIT_L(0); MMA(0, 0, At, B0); BAR; SCHED;
        LDB(B1, 1, 1); STAGE(SB(1, 0), b3);
        BAR; WAIT_L(0); MMA(0, 1, At, B1); BAR;
        LDA(At, 1, 1); STAGE(SA(1, 0), a3);
        BAR; WAIT_L(0); MMA(1, 0, At, B0); BAR; SCHED;
        STAGE(SB(1, 1), b3 + HTB);
        asm volatile("s_waitcnt vmcnt(6)" : "+v"(fl) :: "memory");
        BAR; MMA(1, 1, At, B1); BAR;
        k1 = k3;
    }
    STAMP_AT(3);
    WAIT_V(0);
    if (wr == 0) BAR;

    BAR;
    const bool bad = __builtin_amdgcn_readfirstlane(ld_agent(tmo)) == call;
    {
        const unsigned wo = (unsigned)(wr * 64 + fr) * 512u + (((unsigned)(wc * 8 + fq) ^ (unsigned)(fr & 7)) << 4);
#pragma unroll
        for (int ai = 0; ai < 2; ++ai)
#pragma unroll
            for (int m = 0; m < 4; ++m)
#pragma unroll
                for (int bj = 0; bj < 2; ++bj) {
                    f32x4 v0 = acc[ai][bj][m][0], v1 = acc[ai][bj][m][1];
#pragma unroll
                    for (int e = 0; e < 4; ++e) { v0[e] = v0[e] <= 0.f ? 0.f : v0[e]; v1[e] = v1[e] <= 0.f ? 0.f : v1[e]; }
                    u32x4 w;
                    w.x = cvt_pk_bf16(v0[0], v0[1]); w.y = cvt_pk_bf16(v0[2], v0[3]); w.z = cvt_pk_bf16(v1[0], v1[1]); w.w = cvt_pk_bf16(v1[2], v1[3]);
                    if (bad) w = (u32x4){0x7fc07fc0u, 0x7fc07fc0u, 0x7fc07fc0u, 0x7fc07fc0u};
                    *(LAS u32x4*)(lds + ((wo + (ai * 128 + m * 16) * 512) ^ (bj * 64))) = w;
                }
    }
    __syncthreads();
    {
        const int gr = (wid + cb + rank) & 7;
        const rsrc_t ro = make_rsrc(a.out[(gr + rb) & 7], OUT_BYTES);
        const int hi = lane >> 5, ch = lane & 31;
        const unsigned ooff = ((unsigned)(32 * rb + hi) * NG + (unsigned)(NL * rank + 256 * cb)) * 2u + ch * 16u;
#pragma unroll
        for (int i = 0; i < 16; ++i) {
            const int row = 2 * i + hi;
            const u32x4 w = *(const LAS u32x4*)(lds + (unsigned)(gr * 32 + row) * 512u + ((unsigned)(ch ^ (row & 7)) << 4));
            st16_sys(ro, ooff, i * (2 * NG * 2), w);
#define EPI_THROTTLE 0
            asm volatile("s_waitcnt vmcnt(%c0)" :: "i"(EPI_THROTTLE) : "memory");
        }
    }
    STAMP_AT(4);
    drain();
    __syncthreads();
    if (tid < NR && tid != rank)
        __hip_atomic_fetch_add((gu32*)(a.ws[tid] + OFF_DONE + rank * 128), 1u, __ATOMIC_RELAXED, __HIP_MEMORY_SCOPE_SYSTEM);
    STAMP_AT(5);
}

}

extern "C" void kernel_launch(int rank, int nranks, ncclComm_t comm,
                              hipStream_t stream,
                              void* const* d_in, const int* in_sizes, int n_in,
                              void* d_out, int out_size,
                              const int* in_dtypes, int out_dtype,
                              void* const* d_out_all,
                              void* const* d_ws_all, size_t ws_bytes) {
    if (nranks != NR || (unsigned)rank >= (unsigned)NR || n_in != 2 || ws_bytes < WS_NEED || out_size != ML * NG ||
        in_sizes[0] != 4096 * KL || in_sizes[1] != KL * NG) {
        fprintf(stderr, "kernel_launch: unexpected problem shape\n");
        return;
    }
    HIP_CHECK(hipFuncSetAttribute((const void*)gemm_rs_kernel, hipFuncAttributeMaxDynamicSharedMemorySize, LDS_BYTES));
    Args a;
    a.x = (gbf16*)d_in[0];
    a.w = (gbf16*)d_in[1];
    for (int p = 0; p < NR; ++p) { a.ws[p] = (gchar*)d_ws_all[p]; a.out[p] = (gchar*)d_out_all[p]; }
    a.rank = rank;
    a.pad = 0;
    hipLaunchKernelGGL(gemm_rs_kernel, dim3(GRID), dim3(512), LDS_BYTES, stream, a);
    HIP_CHECK(hipGetLastError());
}
```

```cpp
#include <hip/hip_runtime.h>
#include <rccl/rccl.h>
#include <cstdio>
#include <cstdint>

#define HIP_CHECK(expr) do { hipError_t _e = (expr); if (_e != hipSuccess) { \
    fprintf(stderr, "HIP error %s at %s:%d\n", hipGetErrorString(_e), __FILE__, __LINE__); \
    return; } } while (0)

namespace {

#define LAS __attribute__((address_space(3)))
#define GAS __attribute__((address_space(1)))
typedef GAS char gchar;
typedef GAS unsigned gu32;
typedef GAS const unsigned short gbf16;
typedef short bf16x8 __attribute__((ext_vector_type(8)));
typedef float f32x4 __attribute__((ext_vector_type(4)));
typedef unsigned u32x4 __attribute__((ext_vector_type(4)));

constexpr int NR = 8;
constexpr int KL = 512;
constexpr int NG = 16384;
constexpr int NL = NG / NR;
constexpr int ML = 512;
constexpr int NRB = 16, NCB = 8;
constexpr int NKT = 64;
constexpr unsigned PIECE = 32768;
constexpr int ITEMS_PER_ROUND = NRB + NCB, ITEMS_PER_LINK = 8 * ITEMS_PER_ROUND;

constexpr unsigned UNITS = 4 * ITEMS_PER_LINK;
constexpr unsigned PK_BYTES = 6144;
constexpr unsigned PK_TAIL = 5120, TAIL_BITS = 8128, HDR_BITS = 64;
constexpr unsigned SEQ = 128;
constexpr unsigned OMORE = 2688;
constexpr unsigned OSLOT = 2048;

constexpr size_t OFF_X = 0;
constexpr size_t OFF_W = OFF_X + (size_t)NKT * NRB * PIECE;
constexpr size_t OFF_PK = OFF_W + (size_t)NKT * NCB * PIECE;
constexpr size_t OFF_FLAG = OFF_PK + (size_t)64 * SEQ * PK_BYTES;
constexpr size_t OFF_OFLAG = OFF_FLAG + (size_t)NKT * ITEMS_PER_ROUND * 128;
constexpr size_t OFF_OPK = OFF_OFLAG + (size_t)NR * NRB * NCB * 128;
constexpr size_t OFF_OMORE = OFF_OPK + (size_t)NR * NRB * NCB * 4 * OSLOT;
constexpr size_t OFF_TMO = OFF_OMORE + (size_t)NR * NRB * NCB * 4 * OMORE;
constexpr size_t OFF_QUEUE = OFF_TMO + 128;
constexpr size_t OFF_TICKET = OFF_QUEUE + NR * 128;
constexpr size_t WS_NEED = OFF_TICKET + 256 * 128;

constexpr int NSEND = 63;
constexpr int NUNP = 63;
constexpr int PW = 3;
constexpr int UW = 8;
constexpr int NGEMM = NRB * NCB;
constexpr int NMOVE = NSEND + NUNP;
constexpr int GRID = NMOVE + NGEMM;
constexpr unsigned WPL = NSEND / 7 * PW;
constexpr unsigned RAWN = 9;
static_assert(NSEND % 7 == 0 && NUNP == NSEND && NSEND <= 64 && GRID <= 256 && WPL <= UNITS && PW <= 7 && UW <= 8, "grid");

inline void make_order(int l, unsigned (&tab)[64]) {
    int key[ITEMS_PER_LINK];
    const int due[NRB] = {0, 10, 29, 58, 96, 134, 173, 214, 256, 288, 320, 352, 384, 416, 448, 480};
    const int duec[NCB] = {0, 0, 16, 32, 48, 70, 96, 128};
    for (int j = 0; j < 8; ++j) {
        for (int cb = 0; cb < NCB; ++cb) key[j * ITEMS_PER_ROUND + cb] = 1 << 30;
        for (int rb = 0; rb < NRB; ++rb) {
            const int step = 32 * (7 * j + (13 - l - 2 * rb % 7) % 7);
            key[j * ITEMS_PER_ROUND + NCB + rb] = 2 * (due[rb] + step) + 1;
            for (int cb = 0; cb < NCB; ++cb) {
                const int k = 2 * ((due[rb] > duec[cb] ? due[rb] : duec[cb]) + step);
                if (k < key[j * ITEMS_PER_ROUND + cb]) key[j * ITEMS_PER_ROUND + cb] = k;
            }
        }
    }
    for (int i = 0; i < 64; ++i) tab[i] = 0;
    for (int i = 0; i < ITEMS_PER_LINK; ++i) {
        int pos = 0;
        for (int k = 0; k < ITEMS_PER_LINK; ++k) pos += (key[k] < key[i] || (key[k] == key[i] && k < i)) ? 1 : 0;
        tab[pos >> 2] |= (unsigned)i << (8 * (pos & 3));
    }
}
__device__ __forceinline__ unsigned code_at(unsigned tab, unsigned u) {
    const unsigned w = __builtin_amdgcn_readlane(tab, (u >> 4) & 63u);
    return (((w >> ((u & 12u) * 2u)) & 255u) << 2) | (u & 3u);
}

constexpr unsigned HTB = 16384;
constexpr unsigned LDS_STAGE = 8 * HTB;
constexpr unsigned NSLOT = 2;
constexpr unsigned LDS_BYTES = LDS_STAGE + 1024 + NSLOT * PK_BYTES;
constexpr unsigned long long DEADLINE = 30000000ull;

struct Args {
    gbf16* x;
    gbf16* w;
    gchar* myws;
    int rank;
    gchar* ws[NR];
    gchar* out[NR];
    unsigned order[7][64];
};

__device__ __forceinline__ unsigned cvt_pk_bf16(float lo, float hi) {
    typedef float f32x2 __attribute__((ext_vector_type(2)));
    typedef __bf16 bf16x2 __attribute__((ext_vector_type(2)));
    return __builtin_bit_cast(unsigned, __builtin_convertvector((f32x2){lo, hi}, bf16x2));
}
__device__ __forceinline__ gchar* uniform(gchar* p) {
    const unsigned long long v = (unsigned long long)p;
    const unsigned lo = __builtin_amdgcn_readfirstlane((unsigned)v), hi = __builtin_amdgcn_readfirstlane((unsigned)(v >> 32));
    return (gchar*)(((unsigned long long)hi << 32) | lo);
}
__device__ __forceinline__ void drain() { asm volatile("s_waitcnt vmcnt(0)" ::: "memory"); __builtin_amdgcn_s_waitcnt(0x0F70); }
__device__ __forceinline__ unsigned ld_sys(gu32* p) { return __hip_atomic_load(p, __ATOMIC_RELAXED, __HIP_MEMORY_SCOPE_SYSTEM); }
__device__ __forceinline__ unsigned ld_sys2(gu32* p) {
    const unsigned long long v = __hip_atomic_load((GAS unsigned long long*)p, __ATOMIC_RELAXED, __HIP_MEMORY_SCOPE_SYSTEM);
    return min((unsigned)v, (unsigned)(v >> 32));
}
__device__ __forceinline__ unsigned ld_agent(gu32* p) { return __hip_atomic_load(p, __ATOMIC_RELAXED, __HIP_MEMORY_SCOPE_AGENT); }

__device__ __forceinline__ unsigned wait_flags(gu32* p, unsigned want, gu32* tmo, unsigned call) {
    const unsigned long long t0 = __builtin_amdgcn_s_memrealtime();
    unsigned polls = 0, v;
    while (__ballot((v = ld_sys(p)) < want) != 0ull) {
        if ((++polls & 63u) == 0u &&
            (__builtin_amdgcn_readfirstlane(ld_agent(tmo)) == call || __builtin_amdgcn_s_memrealtime() - t0 > DEADLINE)) {
            __hip_atomic_store(tmo, call, __ATOMIC_RELAXED, __HIP_MEMORY_SCOPE_AGENT);
            break;
        }
        __builtin_amdgcn_s_sleep(2);
    }
    asm volatile("s_waitcnt vmcnt(0) lgkmcnt(0)" ::: "memory");
    __builtin_amdgcn_s_waitcnt(0x0070);
    return __builtin_amdgcn_readfirstlane(v);
}

#define GST(voff, v, sbase, imm) asm volatile("global_store_dwordx4 %0, %1, %2 offset:" #imm " sc0 sc1\n\ts_nop 1" :: "v"(voff), "v"(v), "s"(sbase) : "memory")
#define OST(voff, v, sbase) asm volatile("global_store_dwordx4 %0, %1, %2 sc0 sc1\n\ts_nop 1" :: "v"(voff), "v"(v), "s"(sbase) : "memory")
#define GLDS(g, l) __builtin_amdgcn_global_load_lds((gu32*)(g), (LAS unsigned*)(l), 16, 0, 0)
template <class F>
__device__ __forceinline__ void for_unit(const Args& a, unsigned u, int dst, int lane, F f) {
    const int it = u >> 2, wq = u & 3;
    const int j = it / ITEMS_PER_ROUND, r = it % ITEMS_PER_ROUND, g = lane >> 4, c = lane & 15;
    const bool isw = r < NCB;
    const int rb = r - NCB, g0 = (2 * wq + rb) & 7, g1 = (2 * wq + 1 + rb) & 7;
    gchar* p = isw ? (gchar*)(a.w + (size_t)(64 * j + 32 * (wq & 1)) * NG + NL * dst + 256 * r + 32 * (wq >> 1))
                   : (gchar*)(a.x + (size_t)(ML * g0 + 32 * rb) * KL + 64 * j);
    const long long s0 = isw ? NG * 2 : 64, s1 = isw ? 2 * NG * 2 : 16 * KL * 2, s2 = isw ? 4 * NG * 2 : (g1 - g0) * (ML * KL * 2);
    const unsigned o = isw ? (unsigned)(8 * g * NG + 64 * (c >> 2) + 8 * (c & 3)) * 2u : (unsigned)(c * KL + 8 * g) * 2u;
#pragma unroll
    for (int t = 0; t < 8; ++t) f(t, uniform(p + (t & 1) * s0 + ((t >> 1) & 1) * s1 + (t >> 2) * s2) + o);
}
__device__ __forceinline__ void load_unit(const Args& a, unsigned u, int dst, int lane, LAS unsigned char* buf) {
    for_unit(a, u, dst, lane, [&](int t, gchar* p) { GLDS(p, buf + t * 1024); });
}
__device__ __forceinline__ void load_unit(const Args& a, unsigned u, int dst, int lane, u32x4 (&v)[8]) {
    for_unit(a, u, dst, lane, [&](int t, gchar* p) { v[t] = *(GAS const u32x4*)p; });
}
__device__ __forceinline__ void order_unit(bool isw, const u32x4 (&v)[8], u32x4 (&d)[8]) {
    if (isw) {
#pragma unroll
        for (int i = 0; i < 8; ++i) {
            const unsigned sel = (i & 1) ? 0x07060302u : 0x05040100u;
#pragma unroll
            for (int e = 0; e < 4; ++e) d[i][e] = __builtin_amdgcn_perm(v[2 * e + 1][i >> 1], v[2 * e][i >> 1], sel);
        }
    } else {
#pragma unroll
        for (int t = 0; t < 8; ++t) d[t] = v[t];
    }
}
__device__ __forceinline__ void read_unit(unsigned u, int lane, LAS unsigned char* buf, u32x4 (&d)[8]) {
    u32x4 v[8];
#pragma unroll
    for (int t = 0; t < 8; ++t) v[t] = *(LAS u32x4*)(buf + t * 1024 + lane * 16);
    order_unit((u >> 2) % ITEMS_PER_ROUND < NCB, v, d);
}
#define STORE_HALF(h, d) do { \
    const int it_ = u >> 2, j_ = it_ / ITEMS_PER_ROUND, r_ = it_ % ITEMS_PER_ROUND, kt_ = src * 8 + j_; \
    gchar* p_ = wsd + (r_ < NCB ? OFF_W + (size_t)(kt_ * NCB + r_) * PIECE : OFF_X + (size_t)(kt_ * NRB + (r_ - NCB)) * PIECE) + (u & 3) * 8192 + (h) * 4096; \
    const unsigned o_ = lane * 16u, x_ = r_ < NCB ? 64u : 0u, o1_ = o_ ^ x_, o2_ = o_ ^ (2 * x_), o3_ = o_ ^ (3 * x_); \
    GST(o_, (d)[0], p_, 0); GST(o1_, (d)[1], p_, 1024); GST(o2_, (d)[2], p_, 2048); GST(o3_, (d)[3], p_, 3072); } while (0)
__device__ __forceinline__ void store_raw(int src, unsigned u, gchar* wsd, int lane, const u32x4 (&d)[8]) { STORE_HALF(0, d); STORE_HALF(1, d + 4); }

typedef unsigned short u16x2 __attribute__((ext_vector_type(2)));
typedef short i16x2 __attribute__((ext_vector_type(2)));
__device__ __forceinline__ unsigned pk_max_signed(unsigned a, unsigned b) { return __builtin_bit_cast(unsigned, __builtin_elementwise_max(__builtin_bit_cast(i16x2, a), __builtin_bit_cast(i16x2, b))); }
__device__ __forceinline__ unsigned wave_scan(unsigned v, unsigned& total) {
    unsigned x = v;
    x += __builtin_amdgcn_update_dpp(0u, x, 0x111, 0xf, 0xf, true);
    x += __builtin_amdgcn_update_dpp(0u, x, 0x112, 0xf, 0xf, true);
    x += __builtin_amdgcn_update_dpp(0u, x, 0x114, 0xf, 0xf, true);
    x += __builtin_amdgcn_update_dpp(0u, x, 0x118, 0xf, 0xf, true);
    x += __builtin_amdgcn_update_dpp(0u, x, 0x142, 0xa, 0xf, false);
    x += __builtin_amdgcn_update_dpp(0u, x, 0x143, 0xc, 0xf, false);
    total = __builtin_amdgcn_readlane(x, 63);
    return x - v;
}
__device__ __forceinline__ void make_enc_lut(LAS unsigned char* lut, unsigned id) {
    const unsigned k[4] = {id % 3, id / 3 % 3, id / 9 % 3, id / 27};
    unsigned w = 0, sel = 0x0c0c0c0cu, n2 = 0, n4 = 0;
#pragma unroll
    for (unsigned i = 0; i < 4; ++i) {
        if (k[i]) { w |= (1u << n2) << (8 * i); n2 += 2; }
        if (k[i] == 2) { sel ^= (0x0cu ^ i) << (2 * n4 ^ 8u); n4 += 4; }
    }
    *(LAS u32x4*)(lut + 16 * id) = (u32x4){w, sel, n2 | (n4 << 16), n4};
}
typedef unsigned u32x2 __attribute__((ext_vector_type(2)));
__device__ __forceinline__ void make_dec_lut(LAS unsigned char* lut, unsigned id) {
    unsigned sel = 0x0c0c0c0cu, n = 0, from = id < 16 ? 0x03050700u : 0x06010500u;
#pragma unroll
    for (unsigned i = 0; i < 4; ++i)
        if ((id >> i) & 1u) { sel ^= (0x0cu ^ (from & 255u)) << (8 * i); from >>= 8; ++n; }
    *(LAS u32x2*)(lut + 8 * id) = (u32x2){sel, id < 16 ? 2 * n : 4 * n};
}
struct Packed { u32x4 lo[4], cw, ends; unsigned acc[6], end[6], bits; };
__device__ __forceinline__ void pack_unit(int src, unsigned u, unsigned call, gchar* wsd, int lane, const u32x4 (&d)[8], LAS unsigned char* lut, Packed& k) {
    u32x4 r[8];
#pragma unroll
    for (int t = 0; t < 8; ++t)
#pragma unroll
        for (int e = 0; e < 4; ++e) r[t][e] = __builtin_amdgcn_alignbit(d[t][e], d[t][e], 31);
    unsigned v[16], t[16], sum = 0, total;
#pragma unroll
    for (int g = 0; g < 16; ++g) {
        v[g] = __builtin_amdgcn_perm(r[g >> 1][(g & 1) * 2 + 1], r[g >> 1][(g & 1) * 2], 0x07050301u);
        if (g < 4) sum = __builtin_amdgcn_udot4(v[g], 0x01010101u, sum, false);
    }
    wave_scan(sum, total);
    const unsigned c = min(max((total + 1946u) >> 10, 20u), 253u) - 20u, c4 = c * 0x01010101u;
    u32x4 e[16];
    unsigned nib[16], chk = 0;
#pragma unroll
    for (int g = 0; g < 16; ++g) {
        v[g] -= c4;
        const unsigned s = v[g] ^ 0x10101010u;
        t[g] = __builtin_amdgcn_perm(0xffffd288u, 0x4808d1d0u, s);
        e[g] = *(LAS u32x4*)(size_t)__builtin_amdgcn_udot4(t[g] & 0x18181818u, 0x36120602u, (unsigned)(size_t)lut - 640u, false);
        nib[g] = v[g] & __builtin_amdgcn_perm(0u, 0u, s) & 0x0f0f0f0fu;
        const unsigned w = v[g] - 0x01010101u;
        chk |= w | (w + 0x0a0a0a0au);
    }
    __builtin_amdgcn_sched_barrier(0);
#pragma unroll
    for (int g = 0; g < 16; ++g) k.lo[g >> 2][g & 3] = __builtin_amdgcn_perm(r[g >> 1][(g & 1) * 2 + 1], r[g >> 1][(g & 1) * 2], 0x06040200u);
#pragma unroll
    for (int j = 0; j < 4; ++j)
        k.cw[j] = ((t[4 * j] >> 6) & 0x03030303u) | ((t[4 * j + 1] >> 4) & 0x0c0c0c0cu) | ((t[4 * j + 2] >> 2) & 0x30303030u) | (t[4 * j + 3] & 0xc0c0c0c0u);
    __builtin_amdgcn_sched_barrier(0);
    unsigned cnt[2] = {0u, 0u}, mid[2];
#pragma unroll
    for (int i = 0; i < 6; ++i) k.acc[i] = 0;
#pragma unroll
    for (int g = 0; g < 16; ++g) {
        k.acc[g >> 2] = __builtin_amdgcn_alignbit(__builtin_amdgcn_udot4(t[g] & 0x03030303u, e[g].x, 0u, false), k.acc[g >> 2], e[g].z);
        const unsigned y = __builtin_amdgcn_perm(0u, nib[g], e[g].y);
        k.acc[4 + (g >> 3)] = __builtin_amdgcn_alignbit(__builtin_amdgcn_perm(0u, y | (y << 12), 0x0c0c0301u), k.acc[4 + (g >> 3)], e[g].w);
        cnt[g >> 3] += e[g].z;
        if ((g & 7) == 3) mid[g >> 3] = cnt[g >> 3];
    }
    bool raw = (chk & 0xe0e0e0e0u) != 0u || max(cnt[0], cnt[1]) >= (33u << 16);
    unsigned tot[2], start = HDR_BITS, other;
    const unsigned at[2] = {wave_scan(raw ? 0u : cnt[0], tot[0]), wave_scan(raw ? 0u : cnt[1], tot[1])};
#pragma unroll
    for (int h = 0; h < 2; ++h) {
        const unsigned second = start + (tot[h] & 0xffffu);
        k.end[2 * h] = start + ((at[h] + mid[h]) & 0xffffu);
        k.end[2 * h + 1] = start + ((at[h] + cnt[h]) & 0xffffu);
        k.end[4 + h] = second + ((at[h] + cnt[h]) >> 16);
        other = start;
        start = second + (tot[h] >> 16);
    }
    k.bits = start;
    raw = raw || k.bits > TAIL_BITS;
    const unsigned long long rm = __ballot(raw);
    k.ends = (u32x4){lane == 0 ? 2u * call : 0u, lane == 0 ? c | (u << 9) | (other << 19) : 0u, 0u, 0u};
    if (rm != 0ull) {
        if (raw) {
            store_raw(src, u, wsd, lane, d);
            asm volatile("s_waitcnt vmcnt(0)" ::: "memory");
            k.cw = (u32x4){0u, 0u, 0u, 0u};
#pragma unroll
            for (int i = 0; i < 6; ++i) { k.acc[i] = 0; k.end[i] = 32; }
        }
        k.ends = (u32x4){k.ends.x, lane == 0 ? k.ends.y | 256u : 0u, lane == 63 ? (unsigned)rm : 0u, lane == 63 ? (unsigned)(rm >> 32) : 0u};
        k.bits = 8192;
    }
}
__device__ __forceinline__ void put_slot(LAS unsigned char* slot, int lane, const Packed& k) {
    *(LAS u32x4*)(slot + PK_TAIL + lane * 16) = k.ends;
#pragma unroll
    for (int i = 0; i < 4; ++i) *(LAS u32x4*)(slot + i * 1024 + lane * 16) = k.lo[i];
    *(LAS u32x4*)(slot + 4096 + lane * 16) = k.cw;
#pragma unroll
    for (int i = 0; i < 6; ++i) {
        const unsigned long long e = (unsigned long long)k.acc[i] << (k.end[i] & 31u);
        asm volatile("ds_or_b32 %0, %1\n\tds_or_b32 %0, %2 offset:4"
                     :: "v"((unsigned)(size_t)(slot + PK_TAIL - 4) + (k.end[i] >> 5) * 4u), "v"((unsigned)e), "v"((unsigned)(e >> 32)) : "memory");
    }
}
#define GLDS_SYS(g, l) __builtin_amdgcn_global_load_lds((gu32*)(g), (LAS unsigned*)(l), 16, 0, 17)
__device__ __forceinline__ unsigned stream_at(LAS unsigned char* buf, unsigned at) {
    const LAS unsigned* q = (const LAS unsigned*)(buf + (at >> 5) * 4u);
    return __builtin_amdgcn_alignbit(q[1], q[0], at);
}
__device__ __forceinline__ unsigned unpack_half(int src, gchar* p, unsigned h, gchar* wsd, int lane, LAS unsigned char* buf, LAS unsigned char* lut) {
    gchar *pl = p + h * 2048, *pc = p + 4096 + h * 8;
    u32x4 lo[2], d[4];
    u32x2 cw;
    GLDS_SYS(p + PK_TAIL + lane * 16, buf);
    asm volatile("global_load_dwordx2 %2, %3, %5 sc0 sc1\n\t"
                 "global_load_dwordx4 %0, %3, %4 sc0 sc1\n\t"
                 "global_load_dwordx4 %1, %3, %4 offset:1024 sc0 sc1\n\t"
                 "s_waitcnt vmcnt(0)"
                 : "=&v"(lo[0]), "=&v"(lo[1]), "=&v"(cw) : "v"(lane * 16u), "s"(pl), "s"(pc) : "memory");
    unsigned y[8], at[2], cnt = 0, total;
    u32x2 e[8];
#pragma unroll
    for (int g = 0; g < 8; ++g) {
        y[g] = ((cw[g >> 2] >> (2 * (g & 3))) & 0x03030303u) + 0x01010101u;
        e[g] = *(LAS u32x2*)(size_t)__builtin_amdgcn_udot4(y[g] & 0x04040404u, 0x10080402u, (unsigned)(size_t)lut, false);
    }
    const unsigned hdr = __builtin_amdgcn_readfirstlane(*(LAS unsigned*)(buf + 4)), u = min(hdr >> 9 & 1023u, UNITS - 1u);
    __builtin_amdgcn_sched_barrier(0);
#pragma unroll
    for (int g = 0; g < 8; ++g) { if ((g & 3) == 0) at[g >> 2] = cnt; cnt += e[g].y; }
    const unsigned start = h ? hdr >> 19 : HDR_BITS, first = start + wave_scan(cnt, total), second = start + total;
#pragma unroll
    for (int i = 0; i < 2; ++i) at[i] = stream_at(buf, first + at[i]);
#pragma unroll
    for (int g = 0; g < 8; ++g) {
        const u16x2 a = __builtin_bit_cast(u16x2, at[g >> 2]), a2 = {a.x, a.x};
        const unsigned f = __builtin_amdgcn_perm(__builtin_bit_cast(unsigned, a2 << (u16x2){4, 6}), __builtin_bit_cast(unsigned, a2 << (u16x2){0, 2}), e[g].x);
        at[g >> 2] >>= e[g].y;
        y[g] |= f & 0x03030303u;
        e[g] = *(LAS u32x2*)(size_t)__builtin_amdgcn_udot4((y[g] + 0x01010101u) & 0x08080808u, 0x08040201u, (unsigned)(size_t)lut + 128u, false);
    }
    __builtin_amdgcn_sched_barrier(0);
    cnt = 0;
#pragma unroll
    for (int g = 0; g < 8; ++g) cnt += e[g].y;
    unsigned nb = stream_at(buf, second + wave_scan(cnt, total));
    const unsigned c4 = (hdr & 255u) * 0x01010101u;
#pragma unroll
    for (int g = 0; g < 8; ++g) {
        const unsigned n = __builtin_amdgcn_perm(nb << 4, nb, e[g].x) & 0x0f0f0f0fu, l = lo[g >> 2][g & 3];
        nb >>= e[g].y;
        const unsigned hi = (~__builtin_amdgcn_perm(~0x16151110u, ~0x14131200u, y[g] | ((n + 0x0f0f0f0fu) & 0x10101010u)) | n) + c4;
        const unsigned r0 = __builtin_amdgcn_perm(hi, l, 0x05010400u), r1 = __builtin_amdgcn_perm(hi, l, 0x07030602u);
        d[g >> 1][(g & 1) * 2] = __builtin_amdgcn_alignbit(r0, r0, 1);
        d[g >> 1][(g & 1) * 2 + 1] = __builtin_amdgcn_alignbit(r1, r1, 1);
    }
    if ((hdr & 256u) == 0u || ((*(LAS unsigned*)(buf + 1016 + (lane >> 5) * 4) >> (lane & 31)) & 1u) == 0u) STORE_HALF(h, d);
    return u;
}
__device__ __forceinline__ void make_out_lut(LAS unsigned char* lut, unsigned id, bool sender) {
    const unsigned k[4] = {id % 3, id / 3 % 3, id / 9 % 3, id / 27};
    unsigned long long pk = 0x0c0c0c0c0c0c0c0cull;
    unsigned n = 0, e = 0x0c0c0c0cu, s[2] = {0x0c0c0c0cu, 0x0c0c0c0cu};
#pragma unroll
    for (unsigned i = 0; i < 4; ++i)
        if (k[i]) {
            pk ^= (unsigned long long)(0x0cu ^ (2 * i)) << (8 * n);
            s[i >> 1] ^= (0x0c0cu ^ (n | ((4u + i) << 8))) << (16 * (i & 1));
            ++n;
        }
#pragma unroll
    for (unsigned i = 0; i < 4; ++i)
        if (k[i] == 2) {
            pk ^= (unsigned long long)(0x0cu ^ (2 * i + 1)) << (8 * n);
            e ^= (0x0cu ^ n) << (8 * i);
            ++n;
        }
    if (sender) *(LAS u32x2*)(lut + 8 * id) = (u32x2){(unsigned)pk, (unsigned)(pk >> 32)};
    else *(LAS u32x4*)(lut + 16 * id) = (u32x4){e, s[0], s[1], n};
}
#define GST_ON(on, voff, v, sbase, imm) do { unsigned long long keep_; \
    asm volatile("s_mov_b64 %0, exec\n\ts_mov_b64 exec, %4\n\tglobal_store_dwordx4 %1, %2, %3 offset:" #imm " sc0 sc1\n\ts_mov_b64 exec, %0\n\ts_nop 0" \
                 : "=&s"(keep_) : "v"(voff), "v"(v), "s"(sbase), "s"(on) : "memory"); } while (0)
__device__ __forceinline__ void send_msg(const u32x4 (&d)[4], unsigned top, gchar* wsd, unsigned id, int lane, LAS unsigned char* buf, LAS unsigned char* lut) {
    const unsigned o = lane * 16u, key = (top + 139u) * 0x01010101u;
    *(LAS u32x4*)(buf + 512 + o) = (u32x4){lane == 0 ? top : 0u, 0u, 0u, 0u};
    *(LAS u32x4*)(buf + 1536 + o) = (u32x4){0u, 0u, 0u, 0u};
    unsigned pre[8], cnt = 0;
    u32x2 sel[8], cw = {0u, 0u};
#pragma unroll
    for (int g = 0; g < 8; ++g) {
        const unsigned x0 = d[g >> 1][(g & 1) * 2], x1 = d[g >> 1][(g & 1) * 2 + 1];
        const unsigned e = (key - (__builtin_amdgcn_perm(x1, x0, 0x07050301u) & 0x7f7f7f7fu)) ^ 0x80808080u;
        const unsigned c = (e | __builtin_amdgcn_perm(0x7fffffffu, 0xffffffffu, e)) & ~__builtin_amdgcn_perm(x1, x0, 0x0b0a0908u) & 0x06060606u;
        const unsigned k = __builtin_amdgcn_perm(0x00020001u, 0x00010000u, c);
        sel[g] = *(LAS u32x2*)(size_t)__builtin_amdgcn_udot4(k, 0xd8481808u, (unsigned)(size_t)lut, false);
        pre[g] = cnt;
        cnt = __builtin_amdgcn_udot4(k, 0x01010101u, cnt, false);
        cw[g >> 2] |= (g & 3) ? c << (2 * (g & 3) - 1) : c >> 1;
    }
    *(LAS u32x2*)(buf + lane * 8) = cw;
    unsigned n;
    unsigned m0 = 0u - ((unsigned)(size_t)buf + 516u + wave_scan(cnt, n));
    asm volatile("" : "+v"(m0));
    n += 516;
    if (n > 2048u) {
        *(LAS u32x4*)(buf + 2560 + o) = (u32x4){0u, 0u, 0u, 0u};
        *(LAS u32x4*)(buf + 3584 + o) = (u32x4){0u, 0u, 0u, 0u};
        if (lane == 0) *(LAS u32x4*)(buf + 4608) = (u32x4){0u, 0u, 0u, 0u};
    }
#pragma unroll
    for (int g = 0; g < 8; ++g) {
        const unsigned x0 = d[g >> 1][(g & 1) * 2], x1 = d[g >> 1][(g & 1) * 2 + 1];
        unsigned m = m0 - pre[g];
        asm("" : "+v"(m));
        const unsigned lo = __builtin_amdgcn_perm(x1, x0, sel[g].x), hi = __builtin_amdgcn_perm(x1, x0, sel[g].y);
        LAS unsigned* q = (LAS unsigned*)(size_t)(~m & ~3u);
        __hip_atomic_fetch_or(q, __builtin_amdgcn_alignbyte(lo, 0u, m), __ATOMIC_RELAXED, __HIP_MEMORY_SCOPE_WORKGROUP);
        __hip_atomic_fetch_or(q + 1, __builtin_amdgcn_alignbyte(hi, lo, m), __ATOMIC_RELAXED, __HIP_MEMORY_SCOPE_WORKGROUP);
        __hip_atomic_fetch_or(q + 2, __builtin_amdgcn_alignbyte(0u, hi, m), __ATOMIC_RELAXED, __HIP_MEMORY_SCOPE_WORKGROUP);
    }
    const u32x4 w0 = *(LAS u32x4*)(buf + o), w1 = *(LAS u32x4*)(buf + 1024 + o);
    gchar* slot = wsd + OFF_OPK + (size_t)id * OSLOT;
    const unsigned ol = o & ~31u;
    GST_ON(__ballot(ol < n), o, w0, slot, 0);
    GST_ON(__ballot(ol + 1024u < n) | 1ull, o, w1, slot, 1024);
    if (n > 2048u) {
        gchar* sp = wsd + OFF_OMORE + (size_t)id * OMORE;
#pragma unroll
        for (int i = 0; i < 3; ++i) {
            const unsigned oi = o + i * 1024;
            if (oi + 2048u < n) { const u32x4 w = *(LAS u32x4*)(buf + 2048 + oi); GST(oi, w, sp, 0); }
        }
    }
}
__device__ __forceinline__ void recv_msg(gchar* ws, unsigned id, gchar* outp, unsigned ooff, int lane, LAS unsigned char* buf, LAS unsigned char* lut, gu32* tmo, unsigned call) {
    u32x2 cw;
    unsigned tm;
    gchar* slot = ws + OFF_OPK + (size_t)id * OSLOT;
    GLDS_SYS(slot + lane * 16, buf); GLDS_SYS(slot + 1024 + lane * 16, buf + 1024);
    asm volatile("global_load_dwordx2 %0, %2, %3 sc0 sc1\n\t"
                 "global_load_dword %1, %4, off sc1\n\t"
                 "s_waitcnt vmcnt(0)"
                 : "=&v"(cw), "=&v"(tm) : "v"(lane * 8u), "s"(slot), "v"(tmo) : "memory");
    const bool bad = __builtin_amdgcn_readfirstlane(tm) == call;
    unsigned c[8], pre[8], cnt = 0;
    u32x4 e[8];
#pragma unroll
    for (int g = 0; g < 8; ++g) {
        c[g] = (cw[g >> 2] >> (2 * (g & 3))) & 0x03030303u;
        e[g] = *(LAS u32x4*)(size_t)__builtin_amdgcn_udot4(__builtin_amdgcn_perm(0u, 0x04020200u, c[g]), 0xd8481808u, (unsigned)(size_t)lut, false);
    }
    const unsigned top = *(LAS unsigned*)(buf + 512) & 255u;
    __builtin_amdgcn_sched_barrier(0);
#pragma unroll
    for (int g = 0; g < 8; ++g) { pre[g] = cnt; cnt += e[g].w; }
    unsigned n;
    const unsigned p0 = (unsigned)(size_t)buf + 516u + wave_scan(cnt, n);
    if (n + 516u > 2048u) {
        gchar* more = ws + OFF_OMORE + (size_t)id * OMORE + lane * 16;
        GLDS_SYS(more, buf + 2048); GLDS_SYS(more + 1024, buf + 3072);
        if (lane < (OMORE - 2048) / 16) GLDS_SYS(more + 2048, buf + 4096);
        drain();
    }
    const unsigned tab = (top << 8) | (((top - 1u) & 255u) << 16);
    unsigned w[8][3];
#pragma unroll
    for (int g = 0; g < 8; ++g) {
        pre[g] += p0;
        const LAS unsigned* q = (const LAS unsigned*)(size_t)(pre[g] & ~3u);
        w[g][0] = q[0]; w[g][1] = q[1]; w[g][2] = q[2];
    }
    u32x4 d[4];
#pragma unroll
    for (int g = 0; g < 8; ++g) {
        const unsigned s0 = __builtin_amdgcn_alignbyte(w[g][1], w[g][0], pre[g]), s1 = __builtin_amdgcn_alignbyte(w[g][2], w[g][1], pre[g]);
        const unsigned hi = __builtin_amdgcn_perm(s1, s0, e[g].x) | __builtin_amdgcn_perm(0u, tab, c[g]);
        d[g >> 1][(g & 1) * 2] = __builtin_amdgcn_perm(hi, s0, e[g].y);
        d[g >> 1][(g & 1) * 2 + 1] = __builtin_amdgcn_perm(hi, s0, e[g].z);
    }
    if (bad) {
#pragma unroll
        for (int t = 0; t < 4; ++t) d[t] = (u32x4){0x7fc07fc0u, 0x7fc07fc0u, 0x7fc07fc0u, 0x7fc07fc0u};
    }
#pragma unroll
    for (int t = 0; t < 4; ++t) { const unsigned ot = ooff + (t >> 1) * (16 * NG * 2) + (t & 1) * 64; OST(ot, d[t], outp); }
}
constexpr unsigned OMSGS = NRB * 7 * NCB * 4, OLUT = 8192 + 1536;
__device__ __forceinline__ void make_recv_lut(LAS unsigned char* buf, int lane) {
    make_out_lut(buf + OLUT, lane, false);
    if (lane < 81 - 64) make_out_lut(buf + OLUT, lane + 64, false);
}
__device__ __forceinline__ void recv_msgs(const Args& a, gchar* ws, int bid, int wid, int lane, unsigned call, LAS unsigned char* buf) {
    gu32* tmo = (gu32*)(ws + OFF_TMO);
    make_recv_lut(buf, lane);
    for (unsigned t = wid * NMOVE + bid; t < OMSGS; t += 8 * NMOVE) {
        const unsigned wc = t & 3, cb = (t >> 2) & 7, q = t >> 5, rb = q / 7, src = (a.rank + 1 + q % 7) & 7;
        const unsigned tile = src * (NRB * NCB) + rb * NCB + cb;
        wait_flags((gu32*)(ws + OFF_OFLAG + (size_t)tile * 128 + wc * 4), call, tmo, call);
        const unsigned ooff = ((32 * rb + (lane & 15)) * NG + NL * src + 256 * cb + 64 * wc + 8 * (lane >> 4)) * 2u;
        recv_msg(ws, tile * 4 + wc, a.out[a.rank], ooff, lane, buf, buf + OLUT, tmo, call);
    }
}
#define WAIT_VM(n) asm volatile("s_waitcnt vmcnt(" #n ")" ::: "memory")
__device__ __forceinline__ void draw(unsigned& f, gchar* head) {
    unsigned long long keep;
    asm volatile("s_mov_b64 %1, exec\n\ts_mov_b64 exec, 1\n\tglobal_atomic_add %0, %2, %3, %4 sc0\n\ts_mov_b64 exec, %1"
                 : "=&v"(f), "=&s"(keep) : "v"(0u), "v"(1u), "s"(head) : "memory");
}
__device__ __forceinline__ gu32* flag_of(gchar* ws, int src, unsigned u) { return (gu32*)(ws + OFF_FLAG + (size_t)(src * ITEMS_PER_LINK + (u >> 2)) * 128 + (u & 3) * 8); }
__device__ __forceinline__ void publish_unit(gchar* ws, int src, unsigned u, unsigned call) {
    __hip_atomic_store((GAS unsigned long long*)flag_of(ws, src, u), call * 0x100000001ull, __ATOMIC_RELAXED, __HIP_MEMORY_SCOPE_SYSTEM);
}


__global__ __launch_bounds__(512, 2) void gemm_rs_kernel(const Args a) {
    extern __shared__ __attribute__((aligned(16))) unsigned char smem[];
    LAS unsigned char* lds = (LAS unsigned char*)smem;
    LAS unsigned* ctl = (LAS unsigned*)(lds + LDS_STAGE);
    const int tid = threadIdx.x, wid = __builtin_amdgcn_readfirstlane(tid >> 6), lane = tid & 63;
    const int rank = a.rank;
    int bid = blockIdx.x;
    asm volatile("" : "+s"(bid));
    gchar* ws = a.myws;
    gu32* tmo = (gu32*)(ws + OFF_TMO);
#define code_of(u) code_at(otab, u)
    if (bid < NSEND) {
        if (wid == 7 && lane < 4 + 4 * NSLOT) ctl[lane] = 0;
        const unsigned link = bid % 7;
        const int dst = (rank + 1 + link) & 7;
        gchar* wsd = a.ws[dst];
        LAS unsigned char* buf = lds + wid * 16384;
        LAS unsigned char* slots = lds + LDS_STAGE + 1024;
        volatile LAS unsigned* vctl = ctl;
        if (wid == 0) {
            __builtin_amdgcn_s_barrier();
            unsigned call = 0, prev = ~0u, ends = 0;
            gchar* p = wsd + OFF_PK + (size_t)bid * (SEQ * PK_BYTES);
            for (unsigned head = 0, si = 0; ends < PW; ++head, si = si + 1 == NSLOT ? 0 : si + 1) {
                u32x4 ds = *(volatile LAS u32x4*)(ctl + 4 + 4 * si);
                if (__builtin_amdgcn_readfirstlane(ds.x) != head + 1) {
                    if (prev != ~0u) {
                        drain();
                        if (lane == 0) __hip_atomic_store((gu32*)(p + PK_TAIL), 2u * call, __ATOMIC_RELAXED, __HIP_MEMORY_SCOPE_SYSTEM);
                        prev = ~0u;
                    }
                    do { __builtin_amdgcn_s_sleep(1); ds = *(volatile LAS u32x4*)(ctl + 4 + 4 * si); } while (__builtin_amdgcn_readfirstlane(ds.x) != head + 1);
                }
                const unsigned cc = __builtin_amdgcn_readfirstlane(ds.y), bits = __builtin_amdgcn_readfirstlane(ds.z);
                call = __builtin_amdgcn_readfirstlane(ds.w);
                if (cc >= ~1u) { ends += cc & 1u; if (lane == 0) vctl[1] = head + 1; continue; }
                LAS unsigned char* sl = slots + si * PK_BYTES;
                u32x4 v[6];
#pragma unroll
                for (int i = 0; i < 6; ++i) v[i] = *(volatile LAS u32x4*)(sl + i * 1024 + lane * 16);
                gchar* q = p + 4096;
                const unsigned o = lane * 16u;
                asm volatile("s_waitcnt lgkmcnt(0)" : "+v"(v[0]), "+v"(v[1]), "+v"(v[2]), "+v"(v[3]), "+v"(v[4]), "+v"(v[5]) :: "memory");
                if (lane == 0) vctl[1] = head + 1;
                GST(o, v[0], p, 0); GST(o, v[1], p, 1024); GST(o, v[2], p, 2048);
                WAIT_VM(3);
                GST(o, v[3], p, 3072);
                GST(o, v[4], q, 0);
                GST_ON(__ballot((o & ~31u) * 8u < bits), o, v[5], q, 1024);
                prev = cc;
                p += PK_BYTES;
            }
            drain();
            if (lane <= UW / 2) __hip_atomic_store((gu32*)(p + PK_TAIL + lane * PK_BYTES), 2u * call + (lane != 0), __ATOMIC_RELAXED, __HIP_MEMORY_SCOPE_SYSTEM);
            recv_msgs(a, ws, bid, wid, lane, call, buf);
            return;
        }
        if (wid <= PW) {
            const unsigned otab = a.order[link][lane];
            gchar* head = ws + OFF_QUEUE + link * 128;
            LAS unsigned char* lut = buf + 8192;
            unsigned f, call = 0, cur = bid / 7 + (wid - 1) * (NSEND / 7);
            load_unit(a, code_of(cur), dst, lane, buf);
            draw(f, head);
            make_enc_lut(lut, lane);
            if (lane < 81 - 64) make_enc_lut(lut, lane + 64);
            asm volatile("s_waitcnt lgkmcnt(0)" ::: "memory");
            __builtin_amdgcn_s_barrier();
            unsigned first = cur < RAWN ? ~1u : ~0u;
#define RESERVE(pos) do { unsigned long long keep_; \
                asm volatile("s_mov_b64 %1, exec\n\ts_mov_b64 exec, 1\n\tds_add_rtn_u32 %0, %2, %3\n\ts_mov_b64 exec, %1" \
                             : "=&v"(pos), "=&s"(keep_) : "v"(LDS_STAGE), "v"(1u) : "memory"); } while (0)
#define WAIT_SLOT(pos) while (__builtin_amdgcn_readfirstlane(vctl[1]) + NSLOT <= (pos)) __builtin_amdgcn_s_sleep(2)
#define POST(pos, cc, bits) do { if (lane == 0) *(volatile LAS u32x4*)(ctl + 4 + 4 * ((pos) % NSLOT)) = (u32x4){(pos) + 1, cc, bits, call}; } while (0)
#define RAW_FLAG(cc) do { if (lane == 0) { publish_unit(wsd, rank, cc, call); } } while (0)
            do {
                asm volatile("s_cmp_lt_u32 %1, -2\n\ts_cbranch_scc1 1f\n\ts_waitcnt vmcnt(0)\n1:\n\ts_waitcnt vmcnt(8)"
                             : "+v"(f) : "s"(first) : "scc", "memory");
                const unsigned tk = __builtin_amdgcn_readfirstlane(f), nn = WPL + tk % UNITS, cc = code_of(cur);
                call = tk / UNITS + 1u;
                cur = nn;
                unsigned pos = 0;
                if (first != ~1u) RESERVE(pos);
                u32x4 d[8];
                read_unit(cc, lane, buf, d);
                unsigned gone = vctl[1];
                asm volatile("s_waitcnt lgkmcnt(0)" : "+v"(d[0]), "+v"(d[1]), "+v"(d[2]), "+v"(d[3]), "+v"(d[4]), "+v"(d[5]), "+v"(d[6]), "+v"(d[7]),
                             "+v"(pos), "+v"(gone) :: "memory");
                pos = __builtin_amdgcn_readfirstlane(pos);
                gone = __builtin_amdgcn_readfirstlane(gone);
                if (nn < UNITS) { load_unit(a, code_of(nn), dst, lane, buf); draw(f, head); }
                if (first == ~1u || pos >= SEQ - 16) {
                    store_raw(rank, cc, wsd, lane, d);
                    if (first == ~1u) {
                        first = cc;
                    } else {
                        drain();
                        RAW_FLAG(cc);
                        WAIT_SLOT(pos);
                        POST(pos, ~1u, 0u);
                    }
                    continue;
                }
                Packed k;
                pack_unit(rank, cc, call, wsd, lane, d, lut, k);
                __builtin_amdgcn_sched_barrier(0);
                if (gone + NSLOT <= pos) WAIT_SLOT(pos);
                put_slot(slots + pos % NSLOT * PK_BYTES, lane, k);
                asm volatile("s_waitcnt lgkmcnt(0)" ::: "memory");
                POST(pos, cc, k.bits);
                if (first != ~0u) {
                    if (nn < UNITS) WAIT_VM(9); else WAIT_VM(0);
                    RAW_FLAG(first);
                    first = ~0u;
                }
            } while (cur < UNITS);
            if (first != ~0u) { drain(); RAW_FLAG(first); }
            unsigned pos;
            RESERVE(pos);
            asm volatile("s_waitcnt lgkmcnt(0)" : "+v"(pos) :: "memory");
            pos = __builtin_amdgcn_readfirstlane(pos);
            WAIT_SLOT(pos);
            POST(pos, ~0u, 0u);
            recv_msgs(a, ws, bid, wid, lane, call, buf);
            return;
        }
        asm volatile("s_waitcnt lgkmcnt(0)" ::: "memory");
        __builtin_amdgcn_s_barrier();
    }
    if (bid < NMOVE) {
        const unsigned link = bid % 7;
        unsigned tk;
        draw(tk, ws + OFF_TICKET + (size_t)bid * 128 + wid * 4);
        LAS unsigned char* buf = lds + wid * 16384;
        LAS unsigned char* lut = buf + 8192;
        unsigned u0 = bid < NSEND ? UNITS : 8 * ITEMS_PER_ROUND + wid * NUNP + (bid - NSEND);
        u32x4 v[8];
        if (u0 < UNITS) load_unit(a, u0, rank, lane, v);
        __builtin_amdgcn_sched_barrier(0);
        if (lane < 32) make_dec_lut(lut, lane);
        asm volatile("s_waitcnt vmcnt(0)" : "+v"(tk) :: "memory");
        const unsigned call = __builtin_amdgcn_readfirstlane(tk) + 1u;
        while (u0 < UNITS) {
            u32x4 d[8];
            order_unit((u0 >> 2) % ITEMS_PER_ROUND < NCB, v, d);
            asm volatile("" : "+v"(d[0]), "+v"(d[1]), "+v"(d[2]), "+v"(d[3]), "+v"(d[4]), "+v"(d[5]), "+v"(d[6]), "+v"(d[7]) :: "memory");
            store_raw(rank, u0, ws, lane, d);
            const unsigned u1 = u0 + (u0 < 8 * (ITEMS_PER_ROUND + NUNP) ? 15 - 2 * wid : 8) * NUNP;
            if (u1 < UNITS) { load_unit(a, u1, rank, lane, v); WAIT_VM(8); } else drain();
            if (lane == 0) publish_unit(ws, rank, u0, call);
            u0 = u1;
        }
        const int src = (rank + 7 - link) & 7;
        gchar* p = ws + OFF_PK + (size_t)((bid - NSEND) * SEQ + wid / 2) * PK_BYTES;
        for (unsigned i = (bid < NSEND || wid >= UW) ? SEQ : wid / 2; i + 1 < SEQ; i += UW / 2, p += UW / 2 * PK_BYTES) {
            if (wait_flags((gu32*)(p + PK_BYTES + PK_TAIL), 2u * call, tmo, call) != 2u * call) break;
            const unsigned cc = unpack_half(src, p, wid & 1, ws, lane, buf, lut);
            drain();
            if (lane == 0) {
                __hip_atomic_store(flag_of(ws, src, cc) + (wid & 1), call, __ATOMIC_RELAXED, __HIP_MEMORY_SCOPE_SYSTEM);
            }
        }
        recv_msgs(a, ws, bid, wid, lane, call, buf);
        return;
    }

    unsigned tk;
    draw(tk, ws + OFF_TICKET + (size_t)bid * 128 + wid * 4);
    const int gb = bid - NMOVE;
    const int xcd = bid & 7, idx = gb >> 3;
    const int rb = (idx & 3) * 4 + (xcd >> 1), cb = (xcd & 1) * 4 + (idx >> 2);
    const int wr = wid >> 2, wc = wid & 3, fr = lane & 15, fq = lane >> 4;
    const unsigned flip = (cb & 1) + (cb & 2) * 2;
    u32x4 v0[8];
    {
        const unsigned xo = (lane >> 3) * (KL * 2) + (((lane & 7) ^ (lane >> 4)) << 4), wo = (unsigned)(8 * fq * NG + 8 * fr) * 2u;
        gchar* px = uniform((gchar*)(a.x + (size_t)(ML * (((wid ^ flip) + rb) & 7) + 32 * rb) * KL));
        gchar* pw = uniform((gchar*)(a.w + (size_t)(64 * wr + 32 * (wc & 1)) * NG + NL * rank + 256 * cb + 128 * (wc >> 1)));
#pragma unroll
        for (int t = 0; t < 4; ++t) GLDS(px + t * (8 * KL * 2) + (xo ^ ((t & 1) * 64)), lds + wid * 4096 + t * 1024);
        __builtin_amdgcn_sched_barrier(0);
#pragma unroll
        for (int t = 0; t < 8; ++t) asm volatile("global_load_dwordx4 %0, %1, %2" : "=v"(v0[t]) : "v"(wo), "s"(pw + (size_t)t * (NG * 2)) : "memory");
        __builtin_amdgcn_sched_barrier(0);
#pragma unroll
        for (int t = 0; t < 4; ++t) GLDS(px + t * (8 * KL * 2) + 128 + (xo ^ ((t & 1) * 64)), lds + 2 * HTB + wid * 4096 + t * 1024);
    }
    __builtin_amdgcn_sched_barrier(0);
    LAS unsigned char* lut = lds + LDS_STAGE + 256;
    if (tid >= 384 && tid < 384 + 81) make_out_lut(lut, tid - 384, true);
    const unsigned voff = (unsigned)tid * 16u;
    const unsigned ldsw = (unsigned)wid * 1024u;
    const int aoff = wr * 8192 + lane * 16;
    const int boff = (fr & 3) * 1024 + fq * 256 + ((((wc ^ (fr & 3)) << 2) | (fr >> 2)) << 4);
    const unsigned voffa = voff ^ ((flip & 1) * 4096u);
    const int xh = (flip & 4) ? -(int)HTB : (int)HTB;
    gchar* xbase = ws + OFF_X + (size_t)rb * PIECE + (flip & 4) * (HTB / 4);
    gchar* outp = a.out[rank];
    gchar* wsq[4];
#pragma unroll
    for (int q = 0; q < 4; ++q) {
        wsq[q] = a.ws[((((q >> 1) * 4 + wr * 2 + (q & 1)) ^ flip) + rb) & 7];
        asm volatile("" : "+s"(wsq[q]));
    }
    gchar* wbase = ws + OFF_W + (size_t)cb * PIECE;
    const unsigned fl_r = ((lane & 4) ? (unsigned)cb : (unsigned)(NCB + rb)) * 128u + (lane & 3) * 8u;
#define XP(kt) (xbase + (size_t)(kt) * (NRB * PIECE))
#define WP(kt) (wbase + (size_t)(kt) * (NCB * PIECE))
#define FLAGP(kt) ((gu32*)(ws + OFF_FLAG + (size_t)((kt) * (ITEMS_PER_ROUND * 128u) + fl_r)))
#define SA(b, h) (((b) * 2 + (h)) * HTB)
#define SB(b, h) ((4 + (b) * 2 + (h)) * HTB)
    const unsigned vo[2][2] = {{voff, voff + 8192u}, {voffa, voffa + 8192u}}, ldsb = (unsigned)(size_t)lds + ldsw;
#define STAGE(bufoff, gbase, isx, i) asm volatile("s_add_u32 m0, %2, %c3\n\ts_nop 0\n\tglobal_load_lds_dwordx4 %0, %1 sc0 sc1" \
        :: "v"(vo[isx][i]), "s"(gbase), "s"(ldsb), "i"((bufoff) + (i) * 8192) : "scc", "memory")
#define LDA(dst, b, h) do { _Pragma("unroll") for (int m = 0; m < 4; ++m) _Pragma("unroll") for (int k = 0; k < 2; ++k) \
        dst[m][k] = *(const LAS bf16x8*)(lds + SA(b, h) + aoff + m * 2048 + k * 1024); } while (0)
#define LDB(dst, b, h) do { _Pragma("unroll") for (int n = 0; n < 2; ++n) _Pragma("unroll") for (int k = 0; k < 2; ++k) \
        dst[n][k] = *(const LAS bf16x8*)(lds + SB(b, h) + boff + n * 4096 + k * 8192); } while (0)
#define MMA(ai, bj, At, Bt) do { __builtin_amdgcn_s_setprio(1); \
        _Pragma("unroll") for (int m = 0; m < 4; ++m) _Pragma("unroll") for (int n = 0; n < 2; ++n) _Pragma("unroll") for (int k = 0; k < 2; ++k) \
            acc[ai][bj][m][n] = __builtin_amdgcn_mfma_f32_16x16x32_bf16(Bt[n][k], At[m][k], acc[ai][bj][m][n], 0, 0, 0); \
        __builtin_amdgcn_s_setprio(0); } while (0)
#define WAIT_L(n) asm volatile("s_waitcnt lgkmcnt(" #n ")" ::: "memory")
#define BAR __builtin_amdgcn_s_barrier()
#define SCHED __builtin_amdgcn_sched_barrier(0)

#define PUT(b) do { if (wr != (b)) WAIT_N(12 - 12 * (b)); else { u32x4 d_[8]; \
        asm volatile("s_waitcnt vmcnt(%c8)" : "+v"(v0[0]), "+v"(v0[1]), "+v"(v0[2]), "+v"(v0[3]), "+v"(v0[4]), "+v"(v0[5]), "+v"(v0[6]), "+v"(v0[7]) \
                     : "i"(4 - 4 * (b)) : "memory"); \
        const unsigned o_ = SB(b, 0) + ((wc & 1) + ((fr >> 1) & 2)) * 8192 + fq * 256 + (wc >> 1) * 128 + (fr >> 3) * 64 + (fr & 3) * 16; \
        order_unit(true, v0, d_); \
        _Pragma("unroll") for (int t_ = 0; t_ < 8; ++t_) *(LAS u32x4*)(lds + ((o_ + t_ * 1024) ^ ((t_ & 3) * 64))) = d_[t_]; \
        WAIT_L(0); } } while (0)
#define WAIT_N(n) asm volatile("s_waitcnt vmcnt(%c0)" :: "i"(n) : "memory")
    PUT(0);

    f32x4 acc[2][2][4][2];
#pragma unroll
    for (int i = 0; i < 2; ++i)
#pragma unroll
        for (int j = 0; j < 2; ++j)
#pragma unroll
            for (int m = 0; m < 4; ++m)
#pragma unroll
                for (int n = 0; n < 2; ++n) acc[i][j][m][n] = (f32x4){0.f, 0.f, 0.f, 0.f};
    bf16x8 At[4][2], B0[2][2], B1[2][2];

#define HALF(j, i) do { constexpr int tl_ = (j) >> 2, pt_ = (j) & 3; \
        if (pt_ == 0) STAGE(SB(tl_ & 1, 0), WP(kk[tl_]), 0, i); else if (pt_ == 1) STAGE(SA(tl_ & 1, 0), XP(kk[tl_]), 1, i); \
        else if (pt_ == 2) STAGE(SB(tl_ & 1, 1), WP(kk[tl_]) + HTB, 0, i); else STAGE(SA(tl_ & 1, 1), XP(kk[tl_]) + xh, 1, i); } while (0)
#define PART(j) do { HALF(j, 0); HALF(j, 1); } while (0)
    const unsigned roff = wr * 8192 + fr * 128 + ((fq ^ (fr >> 1)) << 4);
#define LDR(dst, b, h) do { _Pragma("unroll") for (int m = 0; m < 4; ++m) _Pragma("unroll") for (int k = 0; k < 2; ++k) \
        dst[m][k] = *(const LAS bf16x8*)(lds + SA(b, h) + (roff ^ (k * 64)) + m * 2048); } while (0)
#define KPLAIN(b, m2, m3, m4, m6) do { \
        LDB(B0, b, 0); SCHED; LDR(At, b, 0); WAIT_L(8); BAR; WAIT_L(0); MMA(0, 0, At, B0); BAR; SCHED; \
        m2; LDB(B1, b, 1); BAR; WAIT_L(0); MMA(0, 1, At, B1); SCHED; m3; BAR; SCHED; \
        m4; LDR(At, b, 1); BAR; WAIT_L(0); MMA(1, 0, At, B0); BAR; SCHED; \
        m6; BAR; MMA(1, 1, At, B1); } while (0)
    BAR;
    if (wr == 1) BAR;
    KPLAIN(0, , , , PUT(1)); BAR;
    asm volatile("" : "+v"(tk));
    const unsigned call = __builtin_amdgcn_readfirstlane(tk) + 1u;
    u32x2 fl = {0u, 0u};
    unsigned tm = 0, hk = (lane & 56) + ((lane >> 3) == rank ? 4 : 0);
    unsigned long long done = 0;
    volatile LAS unsigned* next = ctl;
    unsigned kk[4];
    kk[0] = rank * 8 + 2; kk[1] = rank * 8 + 3;
#define FLAG_LOAD(taken) do { if (wid == 0) { \
        hk += (lane >> 3) == ((taken) >> 3); \
        done = __ballot((hk >> 3) != (lane >> 3)); \
        asm volatile("global_load_dwordx2 %0, %1, %2 sc0 sc1" : "=v"(fl) : "v"(__umul24(hk, ITEMS_PER_ROUND * 128u) + fl_r), "s"(ws + OFF_FLAG) : "memory"); } } while (0)
    gu32* own3 = FLAGP(rank * 8 + 2 + min(lane >> 3, 2));
    KPLAIN(1,
           if (wid == 0) asm volatile("global_load_dwordx2 %0, %1, off sc0 sc1" : "=v"(fl) : "v"(own3) : "memory"),
           if (wid == 0) {
               asm volatile("s_waitcnt vmcnt(0)" : "+v"(fl) :: "memory");
               if (__ballot(fl.x < call) != 0ull) wait_flags(own3, call, tmo, call);
               if (lane == 0) *next = rank * 8 + 4;
               WAIT_L(0);
           },
           PART(0); PART(1); PART(2); PART(3); PART(4); SCHED,
           if (wr == 1) WAIT_N(4));
    auto whole = [&](unsigned long long n) { n |= done; return (n - 0x0101010101010101ull) & ~n & 0x8080808080808080ull; };
    auto pick = [&]() __attribute__((always_inline)) {
        asm volatile("s_waitcnt vmcnt(6)" : "+v"(fl) :: "memory");
        unsigned long long z = whole(__ballot(min(fl.x, fl.y) < call));
        if (z == 0ull) {
            const unsigned long long t0 = __builtin_amdgcn_s_memrealtime();
            unsigned polls = 0;
            while ((z = whole(__ballot(ld_sys2(FLAGP(hk)) < call))) == 0ull) {
                if ((++polls & 63u) == 0u &&
                    (__builtin_amdgcn_readfirstlane(ld_agent(tmo)) == call || __builtin_amdgcn_s_memrealtime() - t0 > DEADLINE)) {
                    __hip_atomic_store(tmo, call, __ATOMIC_RELAXED, __HIP_MEMORY_SCOPE_AGENT);
                    z = whole(0ull);
                    break;
                }
                __builtin_amdgcn_s_sleep(2);
            }
            asm volatile("s_waitcnt vmcnt(0) lgkmcnt(0)" ::: "memory");
            __builtin_amdgcn_s_waitcnt(0x0070);
        }
        if (lane == __builtin_ctzll(z)) *next = hk;
    };
#define MM(ai, bj, At, Bt, i0, i1) do { _Pragma("unroll") for (int i_ = (i0); i_ < (i1); ++i_) { const int k = i_ >> 3, m = (i_ >> 1) & 3, n = i_ & 1; \
            acc[ai][bj][m][n] = __builtin_amdgcn_mfma_f32_16x16x32_bf16(Bt[n][k], At[m][k], acc[ai][bj][m][n], 0, 0, 0); } } while (0)
#define READ(p) do { constexpr int b_ = (((p) + 1) >> 2) & 1, pt_ = ((p) + 1) & 3; \
        if (pt_ == 0) LDB(B0, b_, 0); else if (pt_ == 1) LDA(At, b_, 0); else if (pt_ == 2) LDB(B1, b_, 1); else LDA(At, b_, 1); } while (0)
#define MULT(p, i0, i1) do { if (((p) & 1) == 0) MM(((p) >> 1) & 1, 0, At, B0, i0, i1); else MM(((p) >> 1) & 1, 1, At, B1, i0, i1); } while (0)
#define PHASE(p, first, n, last, take) do { \
        first; SCHED; \
        if (wr == 0) { last; WAIT_N(n); __builtin_amdgcn_s_setprio(1); BAR; } \
        __builtin_amdgcn_s_waitcnt(0xC07F); \
        SCHED; \
        if (take) { \
            MULT(p, 0, 1); SCHED; const unsigned kv_ = *next; SCHED; MULT(p, 1, 9); SCHED; \
            kk[(p) / 4 + 2] = __builtin_amdgcn_readfirstlane(kv_); \
            asm volatile("" : "+s"(kk[(p) / 4 + 2])); \
            SCHED; MULT(p, 9, 13); \
        } else MULT(p, 0, 13); \
        SCHED; \
        asm volatile("s_cmp_eq_u32 %0, 0" :: "s"(wr) : "scc", "memory"); \
        SCHED; MULT(p, 13, 16); SCHED; \
        asm volatile("s_cbranch_scc1 1f\n\ts_waitcnt vmcnt(%c0)\n\ts_barrier\n1:\n\ts_setprio 0" :: "i"((n) > 2 ? (n) - 2 : 0) : "memory"); \
        SCHED; } while (0)
    {
        if (wr == 0) WAIT_N(6);
        BAR;
        READ(-1);
        for (int t = 2; t < NKT - 2; t += 2) {
#define STEP(p) PHASE(p, READ(p); SCHED; if ((p) % 4 == 3) FLAG_LOAD(kk[(p) / 4 + 2]); PART((p) + 5), 6, \
                      if ((p) % 4 == 1 && wid == 0 && ((p) == 5 || t > 2)) pick(), (p) % 4 == 2)
            STEP(0); STEP(1); STEP(2); STEP(3); STEP(4); STEP(5); STEP(6); STEP(7);
            kk[1] = kk[3];
        }
        asm volatile("global_load_dword %0, %1, off sc1" : "=v"(tm) : "v"(tmo) : "memory");
#define LAST(p) PHASE(p, READ(p); if ((p) < 3) PART((p) < 3 ? (p) + 5 : 7), ((p) < 3 ? 6 : (p) < 5 ? 10 - 2 * (p) : 0), , false)
        LAST(0); LAST(1); LAST(2); LAST(3); LAST(4); LAST(5); LAST(6);
        asm volatile("" : "+v"(tm), "+v"(fl));
        if (wr == 0) BAR;
        MULT(7, 0, 16);
    }
    const bool bad = __builtin_amdgcn_readfirstlane(tm) == call;
    if (bad) {
#pragma unroll
        for (int i = 0; i < 32; ++i) acc[i >> 4][(i >> 3) & 1][(i >> 1) & 3][i & 1] = (f32x4){__builtin_nanf(""), __builtin_nanf(""), __builtin_nanf(""), __builtin_nanf("")};
    }
    __builtin_amdgcn_s_waitcnt(0x0F70);
    {
        LAS unsigned char* buf = lds + (wid & 3) * 8192 + (wid >> 2) * SB(0, 0);
        const unsigned tile = rank * (NRB * NCB) + rb * NCB + cb;
        const unsigned ooff = ((unsigned)(32 * rb + fr) * NG + (unsigned)(NL * rank + 256 * cb + 64 * wc + 8 * fq)) * 2u;
        unsigned top = 0;
        u32x4 own[4] = {};
#pragma unroll
        for (int q = 0; q < 4; ++q) {
            asm volatile("s_setprio %c0" :: "i"(3 - q));
            f32x4 v[8];
            float z = 0.f;
#pragma unroll
            for (int t = 0; t < 8; ++t) {
                v[t] = acc[q >> 1][(t >> 1) & 1][(q & 1) * 2 + (t >> 2)][t & 1];
                z = __builtin_fmaf(v[t][0], v[t][1], z);
                z = __builtin_fmaf(v[t][2], v[t][3], z);
            }
            if (__ballot(z != z) != 0ull) {
#pragma unroll
                for (int t = 0; t < 32; ++t) v[t >> 2][t & 3] = v[t >> 2][t & 3] != v[t >> 2][t & 3] ? __builtin_nanf("") : v[t >> 2][t & 3];
            }
            u32x4 d[4];
#pragma unroll
            for (int t = 0; t < 4; ++t)
                d[t] = (u32x4){cvt_pk_bf16(v[2 * t][0], v[2 * t][1]), cvt_pk_bf16(v[2 * t][2], v[2 * t][3]),
                               cvt_pk_bf16(v[2 * t + 1][0], v[2 * t + 1][1]), cvt_pk_bf16(v[2 * t + 1][2], v[2 * t + 1][3])};
            if (q == 0) {
                unsigned mx = 0;
#pragma unroll
                for (int i = 0; i < 16; ++i) mx = pk_max_signed(mx, d[i >> 2][i & 3]);
                mx = __builtin_amdgcn_readfirstlane(mx);
                top = min(max((max(mx >> 16, mx & 0xffffu) + 0x80u) >> 8, 2u) - 1u, 116u);
            }
            if (wsq[q] == ws) {
#pragma unroll
                for (int t = 0; t < 16; ++t) own[t >> 2][t & 3] = pk_max_signed(d[t >> 2][t & 3], 0u);
            } else {
                send_msg(d, top, wsq[q], tile * 4 + wc, lane, buf, lut);
            }
            __builtin_amdgcn_sched_barrier(0);
        }
        const size_t fo = OFF_OFLAG + (size_t)tile * 128 + wc * 4;
#define OPUB(n, w, i) do { unsigned long long keep_; \
            asm volatile("s_waitcnt vmcnt(" #n ")\n\ts_mov_b64 %0, exec\n\ts_mov_b64 exec, 1\n\tglobal_store_dword %1, %2, %3 sc0 sc1\n\ts_mov_b64 exec, %0" \
                         : "=&s"(keep_) : "v"(0u), "v"(call), "s"((w) + fo) : "memory"); } while (0)
        if (wsq[0] == ws || wsq[1] == ws || wsq[2] == ws || wsq[3] == ws) {
            OPUB(4, wsq[0] == ws ? wsq[1] : wsq[0], 0);
            OPUB(3, wsq[0] == ws || wsq[1] == ws ? wsq[2] : wsq[1], 1);
            OPUB(2, wsq[3] == ws ? wsq[2] : wsq[3], 2);
#pragma unroll
            for (int t = 0; t < 4; ++t) { const unsigned ot = ooff + (t >> 1) * (16 * NG * 2) + (t & 1) * 64; OST(ot, own[t], outp); }
        } else {
            OPUB(6, wsq[0], 0); OPUB(5, wsq[1], 1); OPUB(4, wsq[2], 2); OPUB(3, wsq[3], 3);
        }
    }
}

}

extern "C" void kernel_launch(int rank, int nranks, ncclComm_t comm,
                              hipStream_t stream,
                              void* const* d_in, const int* in_sizes, int n_in,
                              void* d_out, int out_size,
                              const int* in_dtypes, int out_dtype,
                              void* const* d_out_all,
                              void* const* d_ws_all, size_t ws_bytes) {
    if (nranks != NR || (unsigned)rank >= (unsigned)NR || n_in != 2 || ws_bytes < WS_NEED || out_size != ML * NG ||
        in_sizes[0] != 4096 * KL || in_sizes[1] != KL * NG) {
        fprintf(stderr, "kernel_launch: unexpected problem shape\n");
        return;
    }
    HIP_CHECK(hipFuncSetAttribute((const void*)gemm_rs_kernel, hipFuncAttributeMaxDynamicSharedMemorySize, LDS_BYTES));
    Args a;
    a.x = (gbf16*)d_in[0];
    a.w = (gbf16*)d_in[1];
    for (int p = 0; p < NR; ++p) { a.ws[p] = (gchar*)d_ws_all[p]; a.out[p] = (gchar*)d_out_all[p]; }
    a.myws = a.ws[rank];
    a.rank = rank;
    for (int l = 0; l < 7; ++l) make_order(l, a.order[l]);
    hipLaunchKernelGGL(gemm_rs_kernel, dim3(GRID), dim3(512), LDS_BYTES, stream, a);
    HIP_CHECK(hipGetLastError());
}
```
